```python
import numpy as np

kernel_name = "gpu_gemm_ar_m4096_k4096_n4096_w8a8_dqrelu"


ABSMAX_RTOL = 0.001

COMPUTE_DTYPE = "int8"
EPILOGUE = "dequant_relu"
IN_QUANT = "int8"
OUT_QUANT = None
IN_DTYPES = {"x": "int8", "w_mat": "int8",
             "scale_x": "float32", "scale_w": "float32"}
WORLD_SIZE = 8

def setup_inputs():

    x = np.random.randint(-127, 128, size=(4096, 4096)).astype(np.int8)
    w_mat = np.random.randint(-127, 128, size=(4096, 4096)).astype(np.int8)
    scale_x = (0.02 * (1.0 + np.random.rand(1))).astype(np.float32)
    scale_w = (0.02 * (1.0 + np.random.rand(1))).astype(np.float32)
    return {"x": x, "w_mat": w_mat, "scale_x": scale_x, "scale_w": scale_w}

def reference(x, w_mat, scale_x, scale_w):
    acc = x.astype(np.float64) @ w_mat.astype(np.float64)
    y = (acc.astype(np.float32) * (scale_x * scale_w)).astype(np.float32)
    return np.maximum(y, 0.0).astype(np.float32)
```

```cpp
#include <hip/hip_runtime.h>
#include <rccl/rccl.h>
#include <cstdio>

#define HIP_CHECK(expr) do { hipError_t _e = (expr); if (_e != hipSuccess) { \
    fprintf(stderr, "HIP error %s at %s:%d\n", hipGetErrorString(_e), __FILE__, __LINE__); \
    return; } } while (0)

typedef int i32x4 __attribute__((ext_vector_type(4)));
typedef unsigned u32x4 __attribute__((ext_vector_type(4)));
typedef unsigned u32x2 __attribute__((ext_vector_type(2)));
typedef float f32x4 __attribute__((ext_vector_type(4)));
typedef float f32x2 __attribute__((ext_vector_type(2)));
typedef __attribute__((address_space(3))) char* lds_ptr;
typedef __amdgpu_buffer_rsrc_t rsrc_t;

constexpr int NRANK = 8, NSEND = 16, NWG = 256;
constexpr unsigned MN = 4096, KSH = 512;
constexpr unsigned SHARD_BYTES = MN * KSH;
constexpr unsigned UNIT_BYTES = 8192, STEP_BYTES = 64 * UNIT_BYTES, SRC_BYTES = 8 * STEP_BYTES, PAR_BYTES = 8 * SRC_BYTES;
constexpr unsigned CTRL_OFF = 2 * PAR_BYTES;
constexpr unsigned CNT_STRIDE = 128;
constexpr unsigned CALL_REL = 64 * CNT_STRIDE;
constexpr unsigned TMO_REL = CALL_REL + NWG * 64;
constexpr unsigned CTRL_BYTES = TMO_REL + 128;
constexpr unsigned DBG_REL = 65536;
constexpr unsigned long long DEADLINE_TICKS = 50000000ull;
constexpr unsigned STAGE_BYTES = 34 * 1024, STAGE_B = 18 * 1024, SNAP_OFF = 4 * STAGE_BYTES, ABORT_OFF = SNAP_OFF + 4 * 256;
constexpr unsigned XRING_OFF = 16 * UNIT_BYTES;
constexpr unsigned LDS_BYTES = XRING_OFF + 8 * 3072;
#define SEND_WIN 7

#define LDAUX "sc0 sc1"
struct Params {
  const char* x; const char* w; const float* sx; const float* sw; float* out;
  char* me;
  char* peer[14];
  unsigned long long order[NSEND];
  unsigned units[NSEND];
  int rank;
};

#define WAIT_VM(n) asm volatile("s_waitcnt vmcnt(%0)" :: "n"(n) : "memory")
#define WAIT_LGKM(n) asm volatile("s_waitcnt lgkmcnt(%0)" :: "n"(n) : "memory")
#define BARRIER() do { asm volatile("" ::: "memory"); __builtin_amdgcn_s_barrier(); asm volatile("" ::: "memory"); } while (0)
#define SCHED() __builtin_amdgcn_sched_barrier(0)
__device__ __forceinline__ unsigned rfl(unsigned v) { return __builtin_amdgcn_readfirstlane(v); }
__device__ __forceinline__ rsrc_t mk_rsrc(const void* p, unsigned bytes) { return __builtin_amdgcn_make_buffer_rsrc((void*)p, 0, bytes, 0x00020000); }
template <class T> __device__ __forceinline__ T lds_ld(unsigned a) { return *(__attribute__((address_space(3))) const T*)(lds_ptr)(size_t)a; }
template <class T> __device__ __forceinline__ void lds_st(unsigned a, T v) { *(__attribute__((address_space(3))) T*)(lds_ptr)(size_t)a = v; }

__device__ __forceinline__ void st128_sys(u32x4 v, rsrc_t r, unsigned vo, unsigned so) {
  asm volatile("buffer_store_dwordx4 %0, %1, %2, %3 offen sc0 sc1\n\ts_nop 1" :: "v"(v), "v"(vo), "s"(r), "s"(so) : "memory");
}
__device__ __forceinline__ void st128(f32x4 v, rsrc_t r, unsigned vo, unsigned so) {
  asm volatile("buffer_store_dwordx4 %0, %1, %2, %3 offen\n\ts_nop 1" :: "v"(v), "v"(vo), "s"(r), "s"(so) : "memory");
}

#define STAMP(i) do {} while (0)
#define STAMPV(i, v) do {} while (0)

__device__ __forceinline__ void transpose16x8(const u32x2 (&r)[16], u32x4 (&o)[8]) {
#pragma unroll
  for (int d = 0; d < 4; ++d)
#pragma unroll
    for (int h = 0; h < 2; ++h) {
      const unsigned r0 = r[4 * d][h], r1 = r[4 * d + 1][h], r2 = r[4 * d + 2][h], r3 = r[4 * d + 3][h];
      const unsigned t0 = __builtin_amdgcn_perm(r1, r0, 0x05010400u), t1 = __builtin_amdgcn_perm(r1, r0, 0x07030602u);
      const unsigned t2 = __builtin_amdgcn_perm(r3, r2, 0x05010400u), t3 = __builtin_amdgcn_perm(r3, r2, 0x07030602u);
      o[4 * h + 0][d] = __builtin_amdgcn_perm(t2, t0, 0x05040100u);
      o[4 * h + 1][d] = __builtin_amdgcn_perm(t2, t0, 0x07060302u);
      o[4 * h + 2][d] = __builtin_amdgcn_perm(t3, t1, 0x05040100u);
      o[4 * h + 3][d] = __builtin_amdgcn_perm(t3, t1, 0x07060302u);
    }
}
__device__ __forceinline__ void load_w_unit(rsrc_t rw, unsigned wu, unsigned s, unsigned lane, u32x2 (&r)[16]) {
  const unsigned vo = (lane >> 4) * 16 * MN + (lane & 15) * 8, so = s * 64 * MN + wu * 128;
#pragma unroll
  for (int k = 0; k < 16; ++k) r[k] = __builtin_amdgcn_raw_buffer_load_b64(rw, vo, so + k * MN, 0);
}
__device__ __forceinline__ unsigned w_chunk_off(unsigned lane, int j) { return j * 1024 + ((lane ^ ((j >> 2) << 3)) << 4); }

__device__ __forceinline__ void sender(const Params& p, const unsigned ldsb, const unsigned wave, const unsigned lane, unsigned* callp) {
  const unsigned g = blockIdx.x;
  const unsigned call_v = __hip_atomic_load(callp, __ATOMIC_RELAXED, __HIP_MEMORY_SCOPE_AGENT);
  const unsigned long long order = p.order[g];
  const unsigned units = p.units[g], rank = p.rank;
  char* pr[7];
#pragma unroll
  for (int i = 0; i < 7; ++i) pr[i] = p.peer[(g + wave) % 7 + i];
  char* const peer_l = p.peer[lane & 7];
  const rsrc_t rx = mk_rsrc(p.x, SHARD_BYTES), rw = mk_rsrc(p.w, SHARD_BYTES);
  const unsigned vx = (wave * 16 + (lane & 15)) * KSH + (lane >> 4) * 16;
  const unsigned xring = ldsb + XRING_OFF + wave * 3072;
  unsigned pc = 0, cc = 0, pslot = 0, cslot = 0;
  auto xpre = [&](unsigned r) {
    const unsigned code = (unsigned)(order >> (2 * r)) & 3, s = (pc >> (8 * code)) & 7;
    pc += 1u << (8 * code);
    if (code < 2) {
      const unsigned so = ((units >> (8 * code)) & 31) * (128 * KSH) + s * 64;
      asm volatile("s_mov_b32 m0, %0\n\ts_nop 0\n\tbuffer_load_dwordx4 %1, %2, %3 offen lds"
                   :: "s"(xring + pslot), "v"(vx), "s"(rx), "s"(so) : "memory");
    }
    pslot = pslot == 2048 ? 0 : pslot + 1024;
  };
  xpre(0); xpre(1);
  const unsigned call = rfl(call_v) + 1;
  const unsigned doff = (call & 1) * PAR_BYTES + rank * SRC_BYTES;
  rsrc_t rd[7];
#pragma unroll
  for (int i = 0; i < 7; ++i) rd[i] = mk_rsrc(pr[i] + doff, SRC_BYTES);
  unsigned* const flag_l = (unsigned*)(peer_l + CTRL_OFF + rank * 4);
  const unsigned vd = wave * 1024 + lane * 16, base8 = (call - 1) * 8;
  unsigned prev_u = 0, prev_val = 0;
  auto signal = [&](unsigned u, unsigned val) {
    if (lane < 7) __hip_atomic_fetch_max(flag_l + u * (CNT_STRIDE / 4), val, __ATOMIC_RELAXED, __HIP_MEMORY_SCOPE_SYSTEM);
  };
  auto round = [&](unsigned r, auto windowed) {
    constexpr bool WIN = decltype(windowed)::value;
    const unsigned code = (unsigned)(order >> (2 * r)) & 3, s = (cc >> (8 * code)) & 7, u = (units >> (8 * code)) & 63;
    cc += 1u << (8 * code);
    const unsigned la = code < 2 ? xring + cslot : ldsb + ((code & 1) * 8 + s) * UNIT_BYTES + wave * 1024;
    cslot = cslot == 2048 ? 0 : cslot + 1024;
    if (WIN) WAIT_VM(7);
    const u32x4 v = lds_ld<u32x4>(la + lane * 16);
    const unsigned so = s * STEP_BYTES + u * UNIT_BYTES;
#pragma unroll
    for (int i = 0; i < 7; ++i) {
      if (WIN) WAIT_VM(SEND_WIN - 1);
      st128_sys(v, rd[i], vd, so);
    }
    if (WIN) {
      if (SEND_WIN > 7) WAIT_VM(7);
      BARRIER();
      if (wave == (r & 7)) signal(prev_u, prev_val);
    }
    asm volatile("" ::: "memory");
    if (r + 2 < 32) xpre(r + 2);
    prev_u = u; prev_val = base8 + s + 1;
  };
  auto stage_w = [&](u32x2 (&r)[16], unsigned s) {
    asm volatile("s_waitcnt vmcnt(7)" : "+v"(r[0]), "+v"(r[1]), "+v"(r[2]), "+v"(r[3]), "+v"(r[4]), "+v"(r[5]), "+v"(r[6]), "+v"(r[7]) :: "memory");
    asm volatile("" : "+v"(r[8]), "+v"(r[9]), "+v"(r[10]), "+v"(r[11]), "+v"(r[12]), "+v"(r[13]), "+v"(r[14]), "+v"(r[15]) :: "memory");
    u32x4 o[8];
    transpose16x8(r, o);
    const unsigned b = ldsb + ((wave & 1) * 8 + s) * UNIT_BYTES;
#pragma unroll
    for (int j = 0; j < 8; ++j) lds_st<u32x4>(b + w_chunk_off(lane, j), o[j]);
  };
  using T = std::integral_constant<bool, true>; using F = std::integral_constant<bool, false>;
  const unsigned wu = (units >> (16 + 8 * (wave & 1))) & 31;
  u32x2 wr[16];
  WAIT_VM(0);
  round(0, F{});
  unsigned u0 = prev_u, v0 = prev_val;
  load_w_unit(rw, wu, wave >> 1, lane, wr);
  asm volatile("" ::: "memory");
  round(1, F{});
  stage_w(wr, wave >> 1);
  WAIT_LGKM(0);
  BARRIER();
  if (threadIdx.x == 0) __hip_atomic_store(callp, call, __ATOMIC_RELAXED, __HIP_MEMORY_SCOPE_AGENT);
  if (wave == 1) signal(u0, v0);
  u0 = prev_u; v0 = prev_val;
  load_w_unit(rw, wu, 4 + (wave >> 1), lane, wr);
  asm volatile("" ::: "memory");
  round(2, F{});
  stage_w(wr, 4 + (wave >> 1));
  WAIT_LGKM(0);
  BARRIER();
  if (wave == 2) signal(u0, v0);
#pragma unroll 1
  for (unsigned r = 3; r < 32; ++r) round(r, T{});
  WAIT_VM(0);
  BARRIER();
  if (wave == 0) signal(prev_u, prev_val);
  STAMP(1);
}

__device__ __forceinline__ void wait_vm_dyn(unsigned n) {
  asm volatile("s_waitcnt vmcnt(10)\n\ts_cmp_ge_u32 %0, 10\n\ts_cbranch_scc1 .Lwv%=\n\t"
               "s_waitcnt vmcnt(9)\n\ts_cmp_ge_u32 %0, 9\n\ts_cbranch_scc1 .Lwv%=\n\t"
               "s_waitcnt vmcnt(8)\n\ts_cmp_ge_u32 %0, 8\n\ts_cbranch_scc1 .Lwv%=\n\t"
               "s_waitcnt vmcnt(5)\n\ts_cmp_ge_u32 %0, 5\n\ts_cbranch_scc1 .Lwv%=\n\t"
               "s_waitcnt vmcnt(4)\n\ts_cmp_ge_u32 %0, 4\n\ts_cbranch_scc1 .Lwv%=\n\t"
               "s_waitcnt vmcnt(1)\n\ts_cmp_ge_u32 %0, 1\n\ts_cbranch_scc1 .Lwv%=\n\t"
               "s_waitcnt vmcnt(0)\n"
               ".Lwv%=:" :: "s"(n) : "scc", "memory");
}

__global__ __launch_bounds__(512, 2) void fused_gemm_allgather(Params p) {
  extern __shared__ __attribute__((aligned(1024))) char shm[];
  const unsigned ldsb = (unsigned)(size_t)(lds_ptr)shm;
  const unsigned lane = threadIdx.x & 63, wave = rfl(threadIdx.x >> 6);
  char* const me = p.me;
  const unsigned rank = p.rank;
  unsigned* const callp = (unsigned*)(me + CTRL_OFF + CALL_REL + blockIdx.x * 64);
  STAMP(0);
  if (blockIdx.x < NSEND) { sender(p, ldsb, wave, lane, callp); return; }

  const unsigned call_v = __hip_atomic_load(callp, __ATOMIC_RELAXED, __HIP_MEMORY_SCOPE_AGENT);
  const unsigned t = blockIdx.x - NSEND;
  const float scale = *p.sx * *p.sw;
  float* const outp = p.out;
  const unsigned long long t_start = __builtin_amdgcn_s_memrealtime();
  unsigned call;
  if (t < 64) {
    u32x4 o[8];
    unsigned off[2];
    if (t < 32) {
      const rsrc_t rx = mk_rsrc(p.x, SHARD_BYTES);
      const unsigned vo = (lane & 15) * KSH + (lane >> 4) * 16, so = t * (128 * KSH) + wave * 64;
#pragma unroll
      for (int j = 0; j < 8; ++j) o[j] = __builtin_amdgcn_raw_buffer_load_b128(rx, vo, so + j * 16 * KSH, 0);
      off[0] = off[1] = lane * 16;
    } else {
      u32x2 wr[16];
      load_w_unit(mk_rsrc(p.w, SHARD_BYTES), t - 32, wave, lane, wr);
      transpose16x8(wr, o);
      off[0] = lane * 16; off[1] = (lane ^ 8) * 16;
    }
    call = rfl(call_v) + 1;
    const rsrc_t rl = mk_rsrc(me + (call & 1) * PAR_BYTES + rank * SRC_BYTES, SRC_BYTES);
#pragma unroll
    for (int j = 0; j < 8; ++j) st128_sys(o[j], rl, off[j >> 2], wave * STEP_BYTES + t * UNIT_BYTES + j * 1024);
    WAIT_VM(0);
    BARRIER();
    if (threadIdx.x == 0) __hip_atomic_store((unsigned*)(me + CTRL_OFF + t * CNT_STRIDE + rank * 4), call * 8, __ATOMIC_RELAXED, __HIP_MEMORY_SCOPE_SYSTEM);
  } else {
    call = rfl(call_v) + 1;
  }
  const unsigned xcd = t & 7, idx = t >> 3, cg = xcd & 3, rp = xcd >> 2;
  const unsigned ti = idx < 28 ? 2 * (idx >> 2) + rp : 14, tj = cg + 4 * (idx < 28 ? (idx & 3) : 2 * rp + idx - 28);
  const unsigned f0 = ti == 0 ? 0 : 17 * ti + 1;
  const unsigned grp = wave >> 2, wn = wave & 3;
  const bool nine = grp == 0 || ti == 0;
  const unsigned base8 = (call - 1) * 8;
  const rsrc_t rdat = mk_rsrc(me + (call & 1) * PAR_BYTES, PAR_BYTES);
  const rsrc_t rctl = mk_rsrc(me + CTRL_OFF, CTRL_BYTES);
  const unsigned vdma = lane * 16 + (grp == 0 ? (f0 + 4 * wn) * 1024 : 32 * UNIT_BYTES + tj * 2 * UNIT_BYTES + wn * 4096);
  const unsigned mdma = ldsb + (grp == 0 ? 0 : STAGE_B) + wn * 4096;
  const bool extra = wave == 5 || (wave == 6 && ti == 0), poller = wave == 4;
  unsigned per_slot, pol1;
  asm("s_lshr_b32 %0, %2, %3\n\ts_and_b32 %0, %0, 1\n\ts_add_u32 %0, %0, 4\n\ts_lshr_b32 %1, 16, %3\n\ts_and_b32 %1, %1, 1"
      : "=&s"(per_slot), "=&s"(pol1) : "s"(ti == 0 ? 0x60u : 0x20u), "s"(wave) : "scc");
  unsigned vaux;
  if (poller) vaux = (lane & 7) >= 5 ? TMO_REL : ((lane & 7) < 3 ? (f0 >> 3) + (lane & 7) : 29 + 2 * tj + (lane & 7)) * CNT_STRIDE + (lane >> 3) * 4;
  else vaux = lane * 16 + (f0 + 11 + wave) * 1024;
  const unsigned maux = ldsb + (11 + wave) * 1024;
  const unsigned va = ldsb + grp * 9 * 1024 + lane * 16;
  const unsigned vb = ldsb + STAGE_B + (wn >> 1) * UNIT_BYTES + (lane & 1) * 4096 + (lane >> 4) * 256 + ((((wn & 1) * 8 + ((lane & 15) >> 1)) ^ ((lane & 1) * 8)) << 4);
  const unsigned vsnap = ldsb + SNAP_OFF + lane * 4;
  if (threadIdx.x < 256 + 2) lds_st<unsigned>(ldsb + SNAP_OFF + threadIdx.x * 4, threadIdx.x < 256 && (lane & 7) < 5 ? base8 : 0u);
  asm volatile("" :: "s"(call));
  WAIT_LGKM(0);
  BARRIER();
  if (threadIdx.x == 0) __hip_atomic_store(callp, call, __ATOMIC_RELAXED, __HIP_MEMORY_SCOPE_AGENT);

  i32x4 acc[9][4];
#pragma unroll
  for (int i = 0; i < 9; ++i)
#pragma unroll
    for (int j = 0; j < 4; ++j) acc[i][j] = i32x4{0, 0, 0, 0};
  unsigned taken = 0;
  unsigned vmask = 0, left = 64, n = 0, aborted = 0;
  unsigned st = 0, st3 = 3 * STAGE_BYTES;
  auto issue = [&](unsigned c) {
    if (poller)
      asm volatile("s_mov_b32 m0, %0\n\ts_nop 0\n\tbuffer_load_dword %1, %2, 0 offen sc0 sc1 lds"
                   :: "s"(ldsb + SNAP_OFF + (n & 3) * 256), "v"(vaux), "s"(rctl) : "memory");
    unsigned long long m = __builtin_amdgcn_ballot_w64(min(c - base8, 8u) > taken) | 0xE0E0E0E0E0E0E0E0ull;
    m &= m >> 4; m &= m >> 2; m &= m >> 1; m &= 0x0101010101010101ull;
    if (m) {
      const unsigned q8 = __builtin_ctzll(m);
      const unsigned s = __builtin_amdgcn_readlane(taken, q8) & 7;
      taken += (lane & 56) == q8 ? 1u : 0u;
      const unsigned so = (q8 + s) * STEP_BYTES;
      asm volatile("s_nop 4\n\ts_mov_b32 m0, %0\n\ts_nop 0\n\t"
                   "buffer_load_dwordx4 %1, %2, %3 offen " LDAUX " lds\n\t"
                   "buffer_load_dwordx4 %1, %2, %3 offen offset:1024 " LDAUX " lds\n\t"
                   "buffer_load_dwordx4 %1, %2, %3 offen offset:2048 " LDAUX " lds\n\t"
                   "buffer_load_dwordx4 %1, %2, %3 offen offset:3072 " LDAUX " lds"
                   :: "s"(mdma + st3), "v"(vdma), "s"(rdat), "s"(so) : "memory");
      if (extra)
        asm volatile("s_mov_b32 m0, %0\n\ts_nop 0\n\tbuffer_load_dwordx4 %1, %2, %3 offen " LDAUX " lds"
                     :: "s"(maux + st3), "v"(vaux), "s"(rdat), "s"(so) : "memory");
      vmask |= 8;
    }
  };
  auto snap = [&]() -> unsigned {
    unsigned c;
    asm volatile("ds_read_b32 %0, %1" : "=v"(c) : "v"(vsnap + ((n + 2) & 3) * 256) : "memory");
    return c;
  };
  auto advance = [&]() { vmask >>= 1; ++n; st3 = st; st = st == 3 * STAGE_BYTES ? 0 : st + STAGE_BYTES; };
  if (grp == 1) BARRIER();
#pragma unroll 1
  while (true) {
#pragma unroll 1
    while (!(vmask & 1)) {
      unsigned c = snap();
      asm volatile("s_waitcnt lgkmcnt(0)" : "+v"(c) :: "memory");
      issue(c);
      if (vmask == 0) {
        if (__builtin_amdgcn_readlane(c, 7) == call || rfl(lds_ld<unsigned>(ldsb + ABORT_OFF + (n & 1) * 4))) { aborted = 1; left = 1; vmask = 2; }
        if (poller && __builtin_amdgcn_s_memrealtime() - t_start > DEADLINE_TICKS) lds_st<unsigned>(ldsb + ABORT_OFF + ((n + 1) & 1) * 4, 1u);
      }
      wait_vm_dyn(per_slot * (((vmask >> 2) & 1) + ((vmask >> 3) & 1)) + pol1);
      WAIT_LGKM(0);
      BARRIER();
      BARRIER();
      advance();
    }
    unsigned c = snap();
    i32x4 a[9], b[4];
#pragma unroll
    for (int j = 0; j < 4; ++j) b[j] = lds_ld<i32x4>(vb + st + j * 1024);
#pragma unroll
    for (int i = 0; i < 9; ++i) a[i] = lds_ld<i32x4>(va + st + i * 1024);
    SCHED();
    asm volatile("s_waitcnt lgkmcnt(13)" : "+v"(c) :: "memory");
    issue(c);
    wait_vm_dyn(per_slot * (((vmask >> 2) & 1) + ((vmask >> 3) & 1)) + pol1);
    WAIT_LGKM(0);
    SCHED();
    BARRIER();
    SCHED();
    __builtin_amdgcn_s_setprio(1);
#pragma unroll
    for (int i = 0; i < 8; ++i)
#pragma unroll
      for (int j = 0; j < 4; ++j) acc[i][j] = __builtin_amdgcn_mfma_i32_16x16x64_i8(a[i], b[j], acc[i][j], 0, 0, 0);
    if (nine) {
#pragma unroll
      for (int j = 0; j < 4; ++j) acc[8][j] = __builtin_amdgcn_mfma_i32_16x16x64_i8(a[8], b[j], acc[8][j], 0, 0, 0);
    }
    __builtin_amdgcn_s_setprio(0);
    SCHED();
    --left;
    if (left | (grp ^ 1)) BARRIER();
    if (left == 0) break;
    advance();
  }
  STAMP(3);
  STAMPV(5, n);
  STAMPV(7, __builtin_amdgcn_s_memtime());
  unsigned tz = threadIdx.x; asm volatile("" : "+v"(tz));
  const unsigned ln = tz & 63;
  float lo = 0.f;
  if (aborted) {
    if (tz == 0) __hip_atomic_store((unsigned*)(me + CTRL_OFF + TMO_REL), call, __ATOMIC_RELAXED, __HIP_MEMORY_SCOPE_AGENT);
    lo = __builtin_inff();
  }
  const rsrc_t ro = mk_rsrc(outp, MN * MN * 4);
  const unsigned vo = (ln >> 4) * 4 * (MN * 4) + (ln & 15) * 16;
  const unsigned so0 = (f0 + grp * 9) * 16 * (MN * 4) + (tj * 256 + wn * 64) * 4;
#pragma unroll
  for (int i = 0; i < 9; ++i) {
    if (i < 8 || nine) {
#pragma unroll
      for (int r = 0; r < 4; ++r) {
        f32x4 y;
#pragma unroll
        for (int j = 0; j < 4; ++j) y[j] = fmaxf((float)acc[i][j][r] * scale, lo);
        st128(y, ro, vo, so0 + (i * 16 + r) * (MN * 4));
      }
    }
  }
}

static void build_schedule(Params& p) {
  for (int g = 0; g < NSEND; ++g) {
    p.units[g] = (unsigned)g | (unsigned)(g + 16) << 8 | (unsigned)(32 + g) << 16 | (unsigned)(48 + g) << 24;
    unsigned long long o = 0;
    for (int r = 0; r < 32; ++r) o |= (unsigned long long)(r & 3) << (2 * r);
    p.order[g] = o;
  }
}

extern "C" void kernel_launch(int rank, int nranks, ncclComm_t comm,
                              hipStream_t stream,
                              void* const* d_in, const int* in_sizes, int n_in,
                              void* d_out, int out_size,
                              const int* in_dtypes, int out_dtype,
                              void* const* d_out_all,
                              void* const* d_ws_all, size_t ws_bytes) {
  if (nranks != NRANK || (unsigned)rank >= (unsigned)NRANK || n_in != 4 || in_sizes[0] != (int)SHARD_BYTES || in_sizes[1] != (int)SHARD_BYTES ||
      out_size != (int)(MN * MN) || ws_bytes < (size_t)CTRL_OFF + (1u << 20)) {
    fprintf(stderr, "unexpected problem shape\n");
    return;
  }
  HIP_CHECK(hipFuncSetAttribute((const void*)fused_gemm_allgather, hipFuncAttributeMaxDynamicSharedMemorySize, LDS_BYTES));
  Params p;
  p.x = (const char*)d_in[0]; p.w = (const char*)d_in[1]; p.sx = (const float*)d_in[2]; p.sw = (const float*)d_in[3];
  p.out = (float*)d_out;
  p.me = (char*)d_ws_all[rank];
  for (int i = 0; i < 14; ++i) p.peer[i] = (char*)d_ws_all[(rank + 1 + i % 7) % NRANK];
  p.rank = rank;
  build_schedule(p);
  hipLaunchKernelGGL(fused_gemm_allgather, dim3(NWG), dim3(512), LDS_BYTES, stream, p);
  HIP_CHECK(hipGetLastError());
}
```

```cpp
#include <hip/hip_runtime.h>
#include <rccl/rccl.h>
#include <cstdio>

#define HIP_CHECK(expr) do { hipError_t _e = (expr); if (_e != hipSuccess) { \
    fprintf(stderr, "HIP error %s at %s:%d\n", hipGetErrorString(_e), __FILE__, __LINE__); \
    return; } } while (0)

typedef int i32x4 __attribute__((ext_vector_type(4)));
typedef unsigned u32x4 __attribute__((ext_vector_type(4)));
typedef unsigned u32x2 __attribute__((ext_vector_type(2)));
typedef float f32x4 __attribute__((ext_vector_type(4)));
typedef float f32x2 __attribute__((ext_vector_type(2)));
typedef __attribute__((address_space(3))) char* lds_ptr;
typedef __amdgpu_buffer_rsrc_t rsrc_t;

constexpr int NRANK = 8, NSEND = 16, NWG = 256;
constexpr unsigned MN = 4096, KSH = 512;
constexpr unsigned SHARD_BYTES = MN * KSH;
constexpr unsigned UNIT_BYTES = 8192, STEP_BYTES = 64 * UNIT_BYTES, SRC_BYTES = 8 * STEP_BYTES, PAR_BYTES = 8 * SRC_BYTES;
constexpr unsigned CTRL_OFF = 2 * PAR_BYTES;
constexpr unsigned CNT_STRIDE = 128;
constexpr unsigned CALL_REL = 64 * CNT_STRIDE;
constexpr unsigned CTRL_BYTES = CALL_REL + NWG * 64;
constexpr unsigned DBG_REL = 65536;
constexpr unsigned long long DEADLINE_TICKS = 50000000ull;
constexpr unsigned STAGE_BYTES = 35 * 1024, STAGE_B = 18 * 1024, SNAP_OFF = 4 * STAGE_BYTES, CELL_OFF = SNAP_OFF + 256;
constexpr unsigned LDS_BYTES = CELL_OFF + 64;
constexpr unsigned CELL_NONE = 0x80000000u, CELL_ABORT = 0xC0000000u, OOB = 0x40000000u;
constexpr int ROUNDS = 28;
#define SEND_WIN 7
#define LDAUX "sc0 sc1"

struct Params {
  const char* x; const char* w; const float* sx; const float* sw; float* out;
  char* me;
  char* peer[14];
  unsigned short sched[NSEND][32];
  int rank;
};

#define WAIT_VM(n) asm volatile("s_waitcnt vmcnt(%0)" :: "n"(n) : "memory")
#define WAIT_LGKM(n) asm volatile("s_waitcnt lgkmcnt(%0)" :: "n"(n) : "memory")
#define BARRIER() do { asm volatile("" ::: "memory"); __builtin_amdgcn_s_barrier(); asm volatile("" ::: "memory"); } while (0)
#define SCHED() __builtin_amdgcn_sched_barrier(0)
__device__ __forceinline__ unsigned rfl(unsigned v) { return __builtin_amdgcn_readfirstlane(v); }
__device__ __forceinline__ rsrc_t mk_rsrc(const void* p, unsigned bytes) { return __builtin_amdgcn_make_buffer_rsrc((void*)p, 0, bytes, 0x00020000); }
template <class T> __device__ __forceinline__ T lds_ld(unsigned a) { return *(__attribute__((address_space(3))) const T*)(lds_ptr)(size_t)a; }
template <class T> __device__ __forceinline__ void lds_st(unsigned a, T v) { *(__attribute__((address_space(3))) T*)(lds_ptr)(size_t)a = v; }

__device__ __forceinline__ void st128_sys(u32x4 v, rsrc_t r, unsigned vo, unsigned so) {
  asm volatile("buffer_store_dwordx4 %0, %1, %2, %3 offen sc0 sc1\n\ts_nop 1" :: "v"(v), "v"(vo), "s"(r), "s"(so) : "memory");
}
__device__ __forceinline__ void st128(f32x4 v, rsrc_t r, unsigned vo, unsigned so) {
  asm volatile("buffer_store_dwordx4 %0, %1, %2, %3 offen\n\ts_nop 1" :: "v"(v), "v"(vo), "s"(r), "s"(so) : "memory");
}

#define STAMP(i) do {} while (0)
#define STAMPV(i, v) do {} while (0)

__device__ __forceinline__ void transpose16x8(const u32x2 (&r)[16], u32x4 (&o)[8]) {
#pragma unroll
  for (int d = 0; d < 4; ++d)
#pragma unroll
    for (int h = 0; h < 2; ++h) {
      const unsigned r0 = r[4 * d][h], r1 = r[4 * d + 1][h], r2 = r[4 * d + 2][h], r3 = r[4 * d + 3][h];
      const unsigned t0 = __builtin_amdgcn_perm(r1, r0, 0x05010400u), t1 = __builtin_amdgcn_perm(r1, r0, 0x07030602u);
      const unsigned t2 = __builtin_amdgcn_perm(r3, r2, 0x05010400u), t3 = __builtin_amdgcn_perm(r3, r2, 0x07030602u);
      o[4 * h + 0][d] = __builtin_amdgcn_perm(t2, t0, 0x05040100u);
      o[4 * h + 1][d] = __builtin_amdgcn_perm(t2, t0, 0x07060302u);
      o[4 * h + 2][d] = __builtin_amdgcn_perm(t3, t1, 0x05040100u);
      o[4 * h + 3][d] = __builtin_amdgcn_perm(t3, t1, 0x07060302u);
    }
}
__device__ __forceinline__ void load_w_unit(rsrc_t rw, unsigned wu, unsigned s, unsigned lane, u32x2 (&r)[16]) {
  const unsigned vo = (lane >> 4) * 16 * MN + (lane & 15) * 8, so = s * 64 * MN + wu * 128;
#pragma unroll
  for (int k = 0; k < 16; ++k) r[k] = __builtin_amdgcn_raw_buffer_load_b64(rw, vo, so + k * MN, 0);
}

__device__ __forceinline__ unsigned ld_sys(const void* q) { return __hip_atomic_load((const unsigned*)q, __ATOMIC_RELAXED, __HIP_MEMORY_SCOPE_SYSTEM); }

__device__ __forceinline__ void sender(const Params& p, const unsigned ldsb, const unsigned wave, const unsigned lane, unsigned* callp) {
  const unsigned g = blockIdx.x;
  const unsigned call_v = __hip_atomic_load(callp, __ATOMIC_RELAXED, __HIP_MEMORY_SCOPE_AGENT);
  const unsigned ent = p.sched[g][lane & 31];
  const unsigned rank = p.rank;
  char* pr[7];
#pragma unroll
  for (int i = 0; i < 7; ++i) pr[i] = p.peer[(g + wave) % 7 + i];
  char* const peer_l = p.peer[lane & 7];
  const unsigned long long t_start = __builtin_amdgcn_s_memrealtime();
  const unsigned call = rfl(call_v) + 1;
  const unsigned doff = (call & 1) * PAR_BYTES + rank * SRC_BYTES;
  rsrc_t rd[7];
#pragma unroll
  for (int i = 0; i < 7; ++i) rd[i] = mk_rsrc(pr[i] + doff, SRC_BYTES);
  const rsrc_t rs = mk_rsrc(p.me + doff, SRC_BYTES);
  unsigned* const flag_l = (unsigned*)(peer_l + CTRL_OFF + rank * 4);
  const char* const cnt_l = p.me + CTRL_OFF + (ent & 63) * CNT_STRIDE;
  const unsigned vd = wave * 1024 + lane * 16, base8 = (call - 1) * 8;
  const unsigned ring = ldsb + wave * 3072;
  const unsigned entoff = (ent >> 6) * STEP_BYTES + (ent & 63) * UNIT_BYTES;
  while (__builtin_amdgcn_ballot_w64(ld_sys(cnt_l + rank * 4) - base8 < 8u) != 0) {
    if (__builtin_amdgcn_s_memrealtime() - t_start > DEADLINE_TICKS) break;
    __builtin_amdgcn_s_sleep(2);
  }
  STAMP(2);
  unsigned pslot = 0, cslot = 0;
  auto pre = [&](unsigned r) {
    const unsigned so = __builtin_amdgcn_readlane(entoff, r);
    asm volatile("s_nop 4\n\ts_mov_b32 m0, %0\n\ts_nop 0\n\tbuffer_load_dwordx4 %1, %2, %3 offen sc0 sc1 lds"
                 :: "s"(ring + pslot), "v"(vd), "s"(rs), "s"(so) : "memory");
    pslot = pslot == 2048 ? 0 : pslot + 1024;
  };
  pre(0); pre(1);
  BARRIER();
  if (threadIdx.x == 0) __hip_atomic_store(callp, call, __ATOMIC_RELAXED, __HIP_MEMORY_SCOPE_AGENT);
  unsigned first = 0;
  auto signal = [&](unsigned r) {
    const unsigned e = __builtin_amdgcn_readlane(ent, r);
    asm volatile("" : "+v"(first));
    bool ok = __builtin_amdgcn_readlane(first, r) == call;
    while (!ok && (e >> 6) == 7 && __builtin_amdgcn_s_memrealtime() - t_start < DEADLINE_TICKS)
      ok = __builtin_amdgcn_readlane(ld_sys(cnt_l + 32), r) == call;
    if (ok && lane < 7) __hip_atomic_fetch_max(flag_l + (e & 63) * (CNT_STRIDE / 4), base8 + (e >> 6) + 1, __ATOMIC_RELAXED, __HIP_MEMORY_SCOPE_SYSTEM);
  };
  WAIT_VM(1);
#pragma unroll 1
  for (unsigned r = 0; r < ROUNDS; ++r) {
    WAIT_VM(7);
    const u32x4 v = lds_ld<u32x4>(ring + cslot + lane * 16);
    cslot = cslot == 2048 ? 0 : cslot + 1024;
    unsigned so = __builtin_amdgcn_readlane(entoff, r);
    if (wave == (r & 7)) asm volatile("global_load_dword %0, %1, off offset:32 sc0 sc1" : "=v"(first) : "v"(cnt_l) : "memory");
    asm volatile("s_nop 4" : "+s"(so));
#pragma unroll
    for (int i = 0; i < 7; ++i) {
      WAIT_VM(SEND_WIN - 1);
      st128_sys(v, rd[i], vd, so);
    }
    if (SEND_WIN > 7) WAIT_VM(7);
    BARRIER();
    if (wave == (r & 7) && r > 0) signal(r - 1);
    asm volatile("" ::: "memory");
    if (r + 2 < ROUNDS) pre(r + 2);
  }
  WAIT_VM(0);
  BARRIER();
  if (wave == ((ROUNDS - 1) & 7)) signal(ROUNDS - 1);
  STAMP(1);
}

__global__ __launch_bounds__(512, 2) void fused_gemm_allgather(Params p) {
  extern __shared__ __attribute__((aligned(1024))) char shm[];
  const unsigned ldsb = (unsigned)(size_t)(lds_ptr)shm;
  const unsigned lane = threadIdx.x & 63, wave = rfl(threadIdx.x >> 6);
  char* const me = p.me;
  const unsigned rank = p.rank;
  unsigned* const callp = (unsigned*)(me + CTRL_OFF + CALL_REL + blockIdx.x * 64);
  STAMP(0);
  if (blockIdx.x < NSEND) { sender(p, ldsb, wave, lane, callp); return; }

  const unsigned call_v = __hip_atomic_load(callp, __ATOMIC_RELAXED, __HIP_MEMORY_SCOPE_AGENT);
  const unsigned t = blockIdx.x - NSEND;
  const float scale = *p.sx * *p.sw;
  float* const outp = p.out;
  const unsigned long long t_start = __builtin_amdgcn_s_memrealtime();
  unsigned call;
  if (t < 64) {
    u32x4 o[8];
    unsigned off[2];
    if (t < 32) {
      const rsrc_t rx = mk_rsrc(p.x, SHARD_BYTES);
      const unsigned vo = (lane & 15) * KSH + (lane >> 4) * 16, so = t * (128 * KSH) + wave * 64;
#pragma unroll
      for (int j = 0; j < 8; ++j) o[j] = __builtin_amdgcn_raw_buffer_load_b128(rx, vo, so + j * 16 * KSH, 0);
      off[0] = off[1] = lane * 16;
    } else {
      u32x2 wr[16];
      load_w_unit(mk_rsrc(p.w, SHARD_BYTES), t - 32, wave, lane, wr);
      transpose16x8(wr, o);
      off[0] = lane * 16; off[1] = (lane ^ 8) * 16;
    }
    call = rfl(call_v) + 1;
    const unsigned doff = (call & 1) * PAR_BYTES + rank * SRC_BYTES;
    const rsrc_t rl = mk_rsrc(me + doff, SRC_BYTES);
#pragma unroll
    for (int j = 0; j < 8; ++j) st128_sys(o[j], rl, off[j >> 2], wave * STEP_BYTES + t * UNIT_BYTES + j * 1024);
    if (wave == 0) {
#pragma unroll 1
      for (int i = 0; i < 7; ++i) {
        const rsrc_t rp = mk_rsrc(p.peer[t % 7 + i] + doff, SRC_BYTES);
#pragma unroll
        for (int j = 0; j < 8; ++j) st128_sys(o[j], rp, off[j >> 2], t * UNIT_BYTES + j * 1024);
      }
      WAIT_VM(56);
    } else
      WAIT_VM(0);
    BARRIER();
    unsigned* const cnt = (unsigned*)(me + CTRL_OFF + t * CNT_STRIDE);
    if (threadIdx.x == 64) __hip_atomic_store(cnt + rank, call * 8, __ATOMIC_RELAXED, __HIP_MEMORY_SCOPE_SYSTEM);
    if (wave == 0) {
      char* const peer_l = p.peer[lane & 7];
      WAIT_VM(0);
      if (lane < 7) __hip_atomic_fetch_max((unsigned*)(peer_l + CTRL_OFF + t * CNT_STRIDE + rank * 4), (call - 1) * 8 + 1, __ATOMIC_RELAXED, __HIP_MEMORY_SCOPE_SYSTEM);
      if (lane == 7) __hip_atomic_store(cnt + 8, call, __ATOMIC_RELAXED, __HIP_MEMORY_SCOPE_SYSTEM);
    }
  } else {
    call = rfl(call_v) + 1;
  }
  const unsigned xcd = t & 7, idx = t >> 3, cg = xcd & 3, rp = xcd >> 2;
  const unsigned ti = idx < 28 ? 2 * (idx >> 2) + rp : 14, tj = cg + 4 * (idx < 28 ? (idx & 3) : 2 * rp + idx - 28);
  const unsigned f0 = ti == 0 ? 0 : 17 * ti + 1;
  const unsigned grp = wave >> 2, wn = wave & 3;
  const bool nine = grp == 0 || ti == 0;
  const unsigned base8 = (call - 1) * 8;
  const rsrc_t rdat = mk_rsrc(me + (call & 1) * PAR_BYTES, PAR_BYTES);
  const rsrc_t rctl = mk_rsrc(me + CTRL_OFF, CTRL_BYTES);
  const bool poller = wave == 4;
  const unsigned wsrc = 32 * UNIT_BYTES + tj * 2 * UNIT_BYTES;
  const unsigned vdma = lane * 16 + (grp == 0 ? (f0 + 4 * wn) * 1024 : wsrc + (wn - 1) * 4096);
  const unsigned mdma = ldsb + (grp == 0 ? wn * 4096 : STAGE_B + (wn - 1) * 4096);
  const unsigned xslot = wave < 2 ? 16 + wave : wave == 2 ? 30 : wave == 3 ? 34 : 26 + wave;
  const unsigned maux = ldsb + xslot * 1024;
  unsigned vaux;
  if (poller) vaux = ((lane & 7) < 3 ? (f0 >> 3) + (lane & 7) : 32 + 2 * tj + (lane & 1)) * CNT_STRIDE + (lane >> 3) * 4;
  else vaux = lane * 16 + (wave == 3 || (wave == 1 && ti != 0) ? OOB : wave < 2 ? (f0 + xslot) * 1024 : wsrc + (xslot - 18) * 1024);
  const unsigned va = ldsb + grp * 9 * 1024 + lane * 16;
  const unsigned vb = ldsb + STAGE_B + (wn >> 1) * UNIT_BYTES + (lane & 1) * 4096 + (lane >> 4) * 256 + ((((wn & 1) * 8 + ((lane & 15) >> 1)) ^ ((lane & 1) * 8)) << 4);
  unsigned vcell = poller ? ldsb + SNAP_OFF + lane * 4 : ldsb + CELL_OFF;
  unsigned vcw = ldsb + CELL_OFF + 4;
  auto poll = [&]() {
    asm volatile("s_mov_b32 m0, %0\n\ts_nop 0\n\tbuffer_load_dword %1, %2, 0 offen sc0 sc1 lds"
                 :: "s"(ldsb + SNAP_OFF), "v"(vaux), "s"(rctl) : "memory");
  };
  if (poller) { poll(); WAIT_VM(0); }
  if (threadIdx.x == 0) lds_st<unsigned>(ldsb + CELL_OFF, CELL_NONE);
  asm volatile("" :: "s"(call));
  WAIT_LGKM(0);
  BARRIER();
  if (threadIdx.x == 0) __hip_atomic_store(callp, call, __ATOMIC_RELAXED, __HIP_MEMORY_SCOPE_AGENT);

  i32x4 acc[9][4];
#pragma unroll
  for (int i = 0; i < 9; ++i)
#pragma unroll
    for (int j = 0; j < 4; ++j) acc[i][j] = i32x4{0, 0, 0, 0};
  unsigned thr = base8;
  unsigned taken = 0, dnext = CELL_NONE;
  unsigned long long open = 0x0101010101010101ull;
  unsigned vmask = 0, left = 64, aborted = 0;
  unsigned st = 0, st3 = 3 * STAGE_BYTES;
  unsigned ra = va, rb = vb;
  auto ready = [&](unsigned c) -> unsigned long long {
    unsigned long long m = __builtin_amdgcn_ballot_w64(c > thr);
    m &= m >> 4; m &= m >> 2; m &= m >> 1;
    return m & open;
  };
  auto first_read = [&]() -> unsigned {
    unsigned c;
    asm volatile("ds_read_b32 %0, %1" : "=v"(c) : "v"(vcell) : "memory");
    return c;
  };
  auto issue = [&](unsigned c, auto busy, auto is_poller) {
    unsigned d;
    if (decltype(is_poller)::value) {
      d = dnext;
      unsigned long long m = ready(c);
      if (!decltype(busy)::value) {
        while (m == 0 && vmask == 0 && d == CELL_NONE) {
          if (__builtin_amdgcn_s_memrealtime() - t_start > DEADLINE_TICKS) break;
          poll();
          WAIT_VM(0);
          c = first_read();
          asm volatile("s_waitcnt lgkmcnt(0)" : "+v"(c) :: "memory");
          m = ready(c);
        }
      }
      if (m) {
        const unsigned q8 = __builtin_ctzll(m), s = (taken >> (q8 >> 1)) & 7;
        taken += 1u << (q8 >> 1);
        if (s == 7) open &= ~(1ull << q8);
        asm volatile("v_addc_co_u32 %0, vcc, 0, %0, %1" : "+v"(thr) : "s"(0xFFull << q8) : "vcc");
        dnext = (q8 + s) * STEP_BYTES;
      } else {
        dnext = !decltype(busy)::value && vmask == 0 && d == CELL_NONE ? CELL_ABORT : CELL_NONE;
      }
      lds_st<unsigned>(vcw, dnext);
      vcw ^= 4;
      if (open) poll();
    } else {
      d = rfl(c);
      if ((int)d >= 0) {
        asm volatile("s_nop 4\n\ts_mov_b32 m0, %0\n\ts_nop 0\n\t"
                     "buffer_load_dwordx4 %2, %4, %5 offen " LDAUX " lds\n\t"
                     "buffer_load_dwordx4 %2, %4, %5 offen offset:1024 " LDAUX " lds\n\t"
                     "buffer_load_dwordx4 %2, %4, %5 offen offset:2048 " LDAUX " lds\n\t"
                     "buffer_load_dwordx4 %2, %4, %5 offen offset:3072 " LDAUX " lds\n\t"
                     "s_mov_b32 m0, %1\n\ts_nop 0\n\t"
                     "buffer_load_dwordx4 %3, %4, %5 offen " LDAUX " lds"
                     :: "s"(mdma + st3), "s"(maux + st3), "v"(vdma), "v"(vaux), "s"(rdat), "s"(d) : "memory");
      }
    }
    vmask |= (~d >> 28) & 8;
    if (!decltype(busy)::value && d == CELL_ABORT) { aborted = 1; left = 1; vmask = 2; }
    if (!decltype(is_poller)::value) vcell ^= 4;
    const unsigned stn = st == 3 * STAGE_BYTES ? 0 : st + STAGE_BYTES;
    ra = va + stn; rb = vb + stn; st3 = st; st = stn;
    asm volatile("" : "+v"(ra), "+v"(rb), "+v"(vcell));
    if (!decltype(is_poller)::value)
      asm volatile("s_waitcnt vmcnt(10)\n\ts_cmp_ge_u32 %0, 12\n\ts_cbranch_scc1 .Lwv%=\n\t"
                   "s_waitcnt vmcnt(5)\n\ts_cmp_ge_u32 %0, 4\n\ts_cbranch_scc1 .Lwv%=\n\t"
                   "s_waitcnt vmcnt(0)\n"
                   ".Lwv%=:" :: "s"(vmask) : "scc", "memory");
    vmask >>= 1;
  };
  using T = std::integral_constant<bool, true>; using F = std::integral_constant<bool, false>;
  if (grp == 1) BARRIER();
  auto mainloop = [&](auto is_poller) {
#pragma unroll 1
  while (true) {
#pragma unroll 1
    while (!(vmask & 1)) {
      unsigned c = first_read();
      asm volatile("s_waitcnt lgkmcnt(0)" : "+v"(c) :: "memory");
      issue(c, F{}, is_poller);
      WAIT_LGKM(0);
      BARRIER();
      BARRIER();
    }
    unsigned c = first_read();
    i32x4 a[9], b[4];
#pragma unroll
    for (int j = 0; j < 4; ++j) b[j] = lds_ld<i32x4>(rb + j * 1024);
#pragma unroll
    for (int i = 0; i < 9; ++i) a[i] = lds_ld<i32x4>(ra + i * 1024);
    SCHED();
    asm volatile("s_waitcnt lgkmcnt(13)" : "+v"(c) :: "memory");
    issue(c, T{}, is_poller);
    WAIT_LGKM(0);
    SCHED();
    BARRIER();
    SCHED();
    __builtin_amdgcn_s_setprio(1);
#pragma unroll
    for (int i = 0; i < 8; ++i)
#pragma unroll
      for (int j = 0; j < 4; ++j) acc[i][j] = __builtin_amdgcn_mfma_i32_16x16x64_i8(a[i], b[j], acc[i][j], 0, 0, 0);
    if (nine) {
#pragma unroll
      for (int j = 0; j < 4; ++j) acc[8][j] = __builtin_amdgcn_mfma_i32_16x16x64_i8(a[8], b[j], acc[8][j], 0, 0, 0);
    }
    __builtin_amdgcn_s_setprio(0);
    SCHED();
    --left;
    if (left | (grp ^ 1)) BARRIER();
    if (left == 0) break;
  }
  };
  if (poller) mainloop(T{}); else mainloop(F{});
  STAMP(3);
  STAMPV(5, n);
  STAMPV(7, __builtin_amdgcn_s_memtime());
  unsigned tz = threadIdx.x; asm volatile("" : "+v"(tz));
  const unsigned ln = tz & 63;
  float lo = 0.f;
  if (aborted) lo = __builtin_inff();
  const rsrc_t ro = mk_rsrc(outp, MN * MN * 4);
  const unsigned vo = (ln >> 4) * 4 * (MN * 4) + (ln & 15) * 16;
  const unsigned so0 = (f0 + grp * 9) * 16 * (MN * 4) + (tj * 256 + wn * 64) * 4;
#pragma unroll
  for (int i = 0; i < 9; ++i) {
    if (i < 8 || nine) {
#pragma unroll
      for (int r = 0; r < 4; ++r) {
        f32x4 y;
#pragma unroll
        for (int j = 0; j < 4; ++j) y[j] = fmaxf((float)acc[i][j][r] * scale, lo);
        st128(y, ro, vo, so0 + (i * 16 + r) * (MN * 4));
      }
    }
  }
}

static void build_schedule(Params& p) {
  for (int g = 0; g < NSEND; ++g)
    for (int r = 0; r < 32; ++r) {
      const int rr = r < ROUNDS ? r : ROUNDS - 1;
      p.sched[g][r] = (unsigned short)((g + 16 * (rr & 3)) | (1 + (rr >> 2)) << 6);
    }
}

extern "C" void kernel_launch(int rank, int nranks, ncclComm_t comm,
                              hipStream_t stream,
                              void* const* d_in, const int* in_sizes, int n_in,
                              void* d_out, int out_size,
                              const int* in_dtypes, int out_dtype,
                              void* const* d_out_all,
                              void* const* d_ws_all, size_t ws_bytes) {
  if (nranks != NRANK || (unsigned)rank >= (unsigned)NRANK || n_in != 4 || in_sizes[0] != (int)SHARD_BYTES || in_sizes[1] != (int)SHARD_BYTES ||
      out_size != (int)(MN * MN) || ws_bytes < (size_t)CTRL_OFF + (1u << 20)) {
    fprintf(stderr, "unexpected problem shape\n");
    return;
  }
  HIP_CHECK(hipFuncSetAttribute((const void*)fused_gemm_allgather, hipFuncAttributeMaxDynamicSharedMemorySize, LDS_BYTES));
  Params p;
  p.x = (const char*)d_in[0]; p.w = (const char*)d_in[1]; p.sx = (const float*)d_in[2]; p.sw = (const float*)d_in[3];
  p.out = (float*)d_out;
  p.me = (char*)d_ws_all[rank];
  for (int i = 0; i < 14; ++i) p.peer[i] = (char*)d_ws_all[(rank + 1 + i % 7) % NRANK];
  p.rank = rank;
  build_schedule(p);
  hipLaunchKernelGGL(fused_gemm_allgather, dim3(NWG), dim3(512), LDS_BYTES, stream, p);
  HIP_CHECK(hipGetLastError());
}
```

```cpp
#include <hip/hip_runtime.h>
#include <rccl/rccl.h>
#include <cstdio>

#define HIP_CHECK(expr) do { hipError_t _e = (expr); if (_e != hipSuccess) { \
    fprintf(stderr, "HIP error %s at %s:%d\n", hipGetErrorString(_e), __FILE__, __LINE__); \
    return; } } while (0)

typedef int i32x4 __attribute__((ext_vector_type(4)));
typedef unsigned u32x4 __attribute__((ext_vector_type(4)));
typedef unsigned u32x2 __attribute__((ext_vector_type(2)));
typedef float f32x4 __attribute__((ext_vector_type(4)));
typedef float f32x2 __attribute__((ext_vector_type(2)));
typedef __attribute__((address_space(3))) char* lds_ptr;
typedef __amdgpu_buffer_rsrc_t rsrc_t;

constexpr int NRANK = 8, NSEND = 16, NWG = 256;
constexpr unsigned MN = 4096, KSH = 512;
constexpr unsigned SHARD_BYTES = MN * KSH;
constexpr unsigned UNIT_BYTES = 8192, STEP_BYTES = 64 * UNIT_BYTES, SRC_BYTES = 8 * STEP_BYTES, PAR_BYTES = 8 * SRC_BYTES;
constexpr unsigned CTRL_OFF = 2 * PAR_BYTES;
constexpr unsigned CNT_STRIDE = 256;
constexpr unsigned CALL_REL = 64 * CNT_STRIDE;
constexpr unsigned QUEUE_REL = CALL_REL + NWG * 64;
constexpr unsigned CTRL_BYTES = QUEUE_REL + 128;
constexpr unsigned DBG_REL = 65536;
constexpr unsigned long long DEADLINE_TICKS = 50000000ull;
constexpr unsigned STAGE_BYTES = 35 * 1024, STAGE_B = 18 * 1024, SNAP_OFF = 4 * STAGE_BYTES, CELL_OFF = SNAP_OFF + 256;
constexpr unsigned LDS_BYTES = CELL_OFF + 64;
constexpr unsigned CELL_NONE = 0, CELL_ABORT = 0x40000000u, OOB = 0x40000000u;
constexpr unsigned ITEMS = 512, ITEM_END = 8 << 6;
#define PF 4
#define PD (8 - PF)
#define LDAUX "sc0 sc1"

struct Params {
  const char* x; const char* w; const float* sx; const float* sw; float* out;
  char* me;
  char* peer[14];
  alignas(16) unsigned short sched[ITEMS];
  int rank;
};

#define WAIT_VM(n) asm volatile("s_waitcnt vmcnt(%0)" :: "n"(n) : "memory")
#define WAIT_LGKM(n) asm volatile("s_waitcnt lgkmcnt(%0)" :: "n"(n) : "memory")
#define BARRIER() do { asm volatile("" ::: "memory"); __builtin_amdgcn_s_barrier(); asm volatile("" ::: "memory"); } while (0)
#define SCHED() __builtin_amdgcn_sched_barrier(0)
__device__ __forceinline__ unsigned rfl(unsigned v) { return __builtin_amdgcn_readfirstlane(v); }
__device__ __forceinline__ rsrc_t mk_rsrc(const void* p, unsigned bytes) { return __builtin_amdgcn_make_buffer_rsrc((void*)p, 0, bytes, 0x00020000); }
template <class T> __device__ __forceinline__ T lds_ld(unsigned a) { return *(__attribute__((address_space(3))) const T*)(lds_ptr)(size_t)a; }
template <class T> __device__ __forceinline__ void lds_st(unsigned a, T v) { *(__attribute__((address_space(3))) T*)(lds_ptr)(size_t)a = v; }

__device__ __forceinline__ void st128_sys(u32x4 v, rsrc_t r, unsigned vo, unsigned so) {
  asm volatile("buffer_store_dwordx4 %0, %1, %2, %3 offen sc0 sc1\n\ts_nop 1" :: "v"(v), "v"(vo), "s"(r), "s"(so) : "memory");
}
__device__ __forceinline__ void st128(f32x4 v, rsrc_t r, unsigned vo, unsigned so) {
  asm volatile("buffer_store_dwordx4 %0, %1, %2, %3 offen sc1\n\ts_nop 1" :: "v"(v), "v"(vo), "s"(r), "s"(so) : "memory");
}

#define PH(k) do {} while (0)
#define STAMP(i) do {} while (0)
#define STAMPV(i, v) do {} while (0)

__device__ __forceinline__ void transpose16x8(const u32x2 (&r)[16], u32x4 (&o)[8]) {
#pragma unroll
  for (int d = 0; d < 4; ++d)
#pragma unroll
    for (int h = 0; h < 2; ++h) {
      const unsigned r0 = r[4 * d][h], r1 = r[4 * d + 1][h], r2 = r[4 * d + 2][h], r3 = r[4 * d + 3][h];
      const unsigned t0 = __builtin_amdgcn_perm(r1, r0, 0x05010400u), t1 = __builtin_amdgcn_perm(r1, r0, 0x07030602u);
      const unsigned t2 = __builtin_amdgcn_perm(r3, r2, 0x05010400u), t3 = __builtin_amdgcn_perm(r3, r2, 0x07030602u);
      o[4 * h + 0][d] = __builtin_amdgcn_perm(t2, t0, 0x05040100u);
      o[4 * h + 1][d] = __builtin_amdgcn_perm(t2, t0, 0x07060302u);
      o[4 * h + 2][d] = __builtin_amdgcn_perm(t3, t1, 0x05040100u);
      o[4 * h + 3][d] = __builtin_amdgcn_perm(t3, t1, 0x07060302u);
    }
}
__device__ __forceinline__ void load_w_unit(rsrc_t rw, unsigned wu, unsigned s, unsigned lane, u32x2 (&r)[16]) {
  const unsigned vo = (lane >> 4) * 16 * MN + (lane & 15) * 8, so = s * 64 * MN + wu * 128;
#pragma unroll
  for (int k = 0; k < 16; ++k) r[k] = __builtin_amdgcn_raw_buffer_load_b64(rw, vo, so + k * MN, 0);
}

__device__ __forceinline__ void sender(const Params& p, const unsigned ldsb, const unsigned wave, const unsigned lane, unsigned* callp) {
  const unsigned g = blockIdx.x;
  const unsigned call_v = __hip_atomic_load(callp, __ATOMIC_RELAXED, __HIP_MEMORY_SCOPE_AGENT);
  unsigned cur = g, nxt = NSEND + g, prev = 0;
  const unsigned rank = p.rank;
  const rsrc_t rx = mk_rsrc(p.x, SHARD_BYTES);
  const unsigned vd = wave * 1024 + lane * 16, vx = (wave * 16 + (lane & 15)) * KSH + (lane >> 4) * 16;
  unsigned ring = ldsb + wave * 2048;
  const unsigned tab = ldsb + 16384, cell = tab + 2 * ITEMS;
  auto pre_x = [&](unsigned e) {
    asm volatile("s_nop 4\n\ts_mov_b32 m0, %0\n\ts_nop 0\n\tbuffer_load_dwordx4 %1, %2, %3 offen lds"
                 :: "s"(ring), "v"(vx), "s"(rx), "s"((e & 63) * (128 * KSH) + (e >> 6) * 64) : "memory");
  };
  pre_x(cur);
  if (wave == 7) lds_st<u32x4>(tab + lane * 16, *(const u32x4*)&p.sched[lane * 8]);
  char* pr[7];
#pragma unroll
  for (int i = 0; i < 7; ++i) pr[i] = p.peer[(g * 7 / 16 + wave) % 7 + i];
  char* const peer_l = p.peer[lane & 7];
  const unsigned long long t_start = __builtin_amdgcn_s_memrealtime();
  const unsigned call = rfl(call_v) + 1;
  const unsigned doff = (call & 1) * PAR_BYTES + rank * SRC_BYTES;
  rsrc_t rd[7];
#pragma unroll
  for (int i = 0; i < 7; ++i) rd[i] = mk_rsrc(pr[i] + doff, SRC_BYTES);
  const rsrc_t rs = mk_rsrc(p.me + doff, SRC_BYTES);
  char* const flag_l = peer_l + CTRL_OFF + rank * 4;
  const char* const cnt_l = p.me + CTRL_OFF + (32 + (lane & 31)) * CNT_STRIDE + rank * 4;
  const char* const queue = p.me + CTRL_OFF + QUEUE_REL;
  const unsigned base8 = (call - 1) * 8, qbase = (call - 1) * (ITEMS - NSEND) - 2 * NSEND;
  auto signal = [&](unsigned e) {
    const char* const q = flag_l + (e & 63) * CNT_STRIDE + (e >> 6) * 32;
    const unsigned val = base8 + (e >> 6) + 1;
    if (lane < 7) asm volatile("global_store_dword %0, %1, off sc0 sc1" :: "v"(q), "v"(val) : "memory");
  };
  STAMP(2);
  WAIT_VM(0);
  auto loop = [&](auto lead_) {
    constexpr bool lead = decltype(lead_)::value;
    constexpr int X = lead ? 3 : 1;
    unsigned drawn = 0, chk = base8, one = 1;
    bool staged = false;
#pragma unroll 1
    for (unsigned r = 0;; ++r) {
      WAIT_VM(4);
      const u32x4 v = lds_ld<u32x4>(ring + lane * 16);
      ring ^= 1024;
      unsigned so = (cur >> 6) * STEP_BYTES + (cur & 63) * UNIT_BYTES;
      asm volatile("s_nop 4" : "+s"(so));
#pragma unroll
      for (int i = 0; i < 3; ++i) { WAIT_VM(3); st128_sys(v, rd[i], vd, so); }
      WAIT_VM(3);
      if (lead && r > 0) {
        asm volatile("" : "+v"(drawn), "+v"(chk));
        const unsigned idx = rfl(drawn) - qbase;
        nxt = idx < ITEMS ? rfl(lds_ld<unsigned short>(tab + 2 * idx)) : ITEM_END;
        if (!staged) {
          while ((nxt & 32) && __builtin_amdgcn_ballot_w64(chk - base8 < 8u) != 0 && __builtin_amdgcn_s_memrealtime() - t_start < DEADLINE_TICKS)
            asm volatile("global_load_dword %0, %1, off sc0 sc1\n\ts_waitcnt vmcnt(0)" : "+v"(chk) : "v"(cnt_l) : "memory");
          staged = __builtin_amdgcn_ballot_w64(chk - base8 < 8u) == 0;
        }
        lds_st<unsigned>(cell + (r & 1) * 4, nxt);
        WAIT_LGKM(0);
      }
      BARRIER();
      if (lead) {
        if (r > 0) signal(prev);
        else if (lane == 0) asm volatile("global_store_dword %0, %1, off sc1" :: "v"(callp), "v"(call) : "memory");
        if (nxt != ITEM_END && lane == 0) asm volatile("global_atomic_add %0, %1, %2, off sc0 sc1" : "+v"(drawn) : "v"(queue), "v"(one) : "memory");
        if (!staged) asm volatile("global_load_dword %0, %1, off sc0 sc1" : "+v"(chk) : "v"(cnt_l) : "memory");
      } else if (r > 0) {
        nxt = rfl(lds_ld<unsigned>(cell + (r & 1) * 4));
      }
      if (!(nxt & 32)) pre_x(nxt);
      else asm volatile("s_nop 4\n\ts_mov_b32 m0, %0\n\ts_nop 0\n\tbuffer_load_dwordx4 %1, %2, %3 offen sc0 sc1 lds"
                        :: "s"(ring), "v"(vd), "s"(rs), "s"((nxt >> 6) * STEP_BYTES + (nxt & 63) * UNIT_BYTES) : "memory");
      st128_sys(v, rd[3], vd, so);
#pragma unroll
      for (int i = 4; i < 7; ++i) { if (nxt != ITEM_END && r >= 1) WAIT_VM(3 + X); st128_sys(v, rd[i], vd, so); }
      prev = cur; cur = nxt;
      if (cur == ITEM_END) break;
    }
    WAIT_VM(0);
    BARRIER();
    if (lead) signal(prev);
  };
  if (wave == 7) loop(std::integral_constant<bool, true>{}); else loop(std::integral_constant<bool, false>{});
  STAMP(1);
}

__global__ __launch_bounds__(512, 2) void fused_gemm_allgather(Params p) {
  extern __shared__ __attribute__((aligned(1024))) char shm[];
  const unsigned ldsb = (unsigned)(size_t)(lds_ptr)shm;
  const unsigned lane = threadIdx.x & 63, wave = rfl(threadIdx.x >> 6);
  char* const me = p.me;
  const unsigned rank = p.rank;
  unsigned* const callp = (unsigned*)(me + CTRL_OFF + CALL_REL + blockIdx.x * 64);
  STAMP(0);
  if (blockIdx.x < NSEND) { sender(p, ldsb, wave, lane, callp); return; }

  const unsigned call_v = __hip_atomic_load(callp, __ATOMIC_RELAXED, __HIP_MEMORY_SCOPE_AGENT);
  const unsigned t = blockIdx.x - NSEND;
  const float scale = *p.sx * *p.sw;
  float* const outp = p.out;
  const unsigned long long t_start = __builtin_amdgcn_s_memrealtime();
  unsigned call;
  if (t < 64) {
    u32x4 o[8];
    unsigned off[2];
    if (t < 32) {
      const rsrc_t rx = mk_rsrc(p.x, SHARD_BYTES);
      const unsigned vo = (lane & 15) * KSH + (lane >> 4) * 16, so = t * (128 * KSH) + wave * 64;
#pragma unroll
      for (int j = 0; j < 8; ++j) o[j] = __builtin_amdgcn_raw_buffer_load_b128(rx, vo, so + j * 16 * KSH, 0);
      off[0] = off[1] = lane * 16;
    } else {
      u32x2 wr[16];
      load_w_unit(mk_rsrc(p.w, SHARD_BYTES), t - 32, wave, lane, wr);
      transpose16x8(wr, o);
      off[0] = lane * 16; off[1] = (lane ^ 8) * 16;
    }
    call = rfl(call_v) + 1;
    const unsigned doff = (call & 1) * PAR_BYTES + rank * SRC_BYTES;
    const rsrc_t rl = mk_rsrc(me + doff, SRC_BYTES);
#pragma unroll
    for (int j = 0; j < 8; ++j) st128_sys(o[j], rl, off[j >> 2], wave * STEP_BYTES + t * UNIT_BYTES + j * 1024);
    WAIT_VM(0);
    BARRIER();
    if (threadIdx.x < 8) __hip_atomic_store((unsigned*)(me + CTRL_OFF + t * CNT_STRIDE) + threadIdx.x * 8 + rank, call * 8, __ATOMIC_RELAXED, __HIP_MEMORY_SCOPE_SYSTEM);
  } else {
    call = rfl(call_v) + 1;
  }
  const unsigned xcd = t & 7, idx = t >> 3, cg = xcd & 3, rp = xcd >> 2;
  const unsigned ti = idx < 28 ? 2 * (idx >> 2) + rp : 14, tj = 4 * cg + (idx < 28 ? (idx & 3) : 2 * rp + idx - 28);
  const unsigned f0 = 16 * ti;
  const unsigned grp = wave >> 2, wn = wave & 3;
  const bool nine = grp == 0 || ti == 14;
  const unsigned base8 = (call - 1) * 8;
  const rsrc_t rdat = mk_rsrc(me + (call & 1) * PAR_BYTES, PAR_BYTES);
  const rsrc_t rctl = mk_rsrc(me + CTRL_OFF, CTRL_BYTES);
  const bool poller = wave == 4;
  const unsigned wsrc = 32 * UNIT_BYTES + tj * 2 * UNIT_BYTES;
  const unsigned vdma = lane * 16 + (grp == 0 ? (f0 + 4 * wn) * 1024 : wsrc + (wn - 1) * 4096);
  const unsigned mdma = ldsb + (grp == 0 ? wn * 4096 : STAGE_B + (wn - 1) * 4096);
  const unsigned xslot = wave < 2 ? 16 + wave : wave == 2 ? 30 : wave == 3 ? 34 : 26 + wave;
  const unsigned maux = ldsb + xslot * 1024;
  unsigned vaux;
  if (poller) vaux = ((lane & 7) < 2 ? 2 * ti + (lane & 1) : (lane & 7) == 2 ? 30 + (ti >> 3) : 32 + 2 * tj + (lane & 1)) * CNT_STRIDE + (lane >> 3) * 4;
  else vaux = lane * 16 + (wave == 3 || (wave == 1 && ti != 14) ? OOB : wave < 2 ? (240 + ti + wave) * 1024 : wsrc + (xslot - 18) * 1024);
  const unsigned vpoll = vaux;
  const unsigned va = ldsb + grp * 9 * 1024 + lane * 16;
  const unsigned vb = ldsb + STAGE_B + (wn >> 1) * UNIT_BYTES + (lane & 1) * 4096 + (lane >> 4) * 256 + ((((wn & 1) * 8 + ((lane & 15) >> 1)) ^ ((lane & 1) * 8)) << 4);
  unsigned vcell = poller ? ldsb + SNAP_OFF + lane * 4 : ldsb + CELL_OFF;
  unsigned vcw = ldsb + CELL_OFF + 4;
  auto poll = [&]() {
    asm volatile("s_mov_b32 m0, %0\n\ts_nop 0\n\tbuffer_load_dword %1, %2, 0 offen sc0 sc1 lds"
                 :: "s"(ldsb + SNAP_OFF), "v"(vaux), "s"(rctl) : "memory");
  };
  if (poller) { poll(); WAIT_VM(0); }
  if (threadIdx.x == 0) lds_st<unsigned>(ldsb + CELL_OFF, CELL_NONE);
  asm volatile("" :: "s"(call));
  WAIT_LGKM(0);
  BARRIER();
  if (threadIdx.x == 0) __hip_atomic_store(callp, call, __ATOMIC_RELAXED, __HIP_MEMORY_SCOPE_AGENT);

  i32x4 acc[9][4];
#pragma unroll
  for (int i = 0; i < 9; ++i)
#pragma unroll
    for (int j = 0; j < 4; ++j) acc[i][j] = i32x4{0, 0, 0, 0};
  unsigned thr = base8;
  unsigned taken = 0, dnext = CELL_NONE;
  unsigned long long open = 0x0101010101010101ull;
  unsigned ql = 0, left = 64, aborted = 0;
  unsigned st = 0, stt = 0;
  unsigned ra = va, rb = vb;
  auto ready = [&](unsigned c) -> unsigned long long {
    unsigned long long m = __builtin_amdgcn_ballot_w64(c > thr);
    m &= m >> 4; m &= m >> 2; m &= m >> 1;
    return m & open;
  };
  auto first_read = [&]() -> unsigned {
    unsigned c;
    asm volatile("ds_read_b32 %0, %1" : "=v"(c) : "v"(vcell) : "memory");
    return c;
  };
  auto issue = [&](unsigned c, auto busy, auto is_poller) {
    unsigned d;
    if (decltype(is_poller)::value) {
      d = dnext;
      ql += __builtin_popcount(d & 0x808080u);
      unsigned long long m = ready(c);
      bool dead = false;
      if (!decltype(busy)::value) {
        while (m == 0 && ql == 0 && d != CELL_ABORT) {
          if (__builtin_amdgcn_s_memrealtime() - t_start > DEADLINE_TICKS) { dead = true; open = 0; break; }
          poll();
          WAIT_VM(0);
          c = first_read();
          asm volatile("s_waitcnt lgkmcnt(0)" : "+v"(c) :: "memory");
          m = ready(c);
        }
      }
      dnext = dead ? CELL_ABORT : CELL_NONE;
      unsigned sh = 0;
      const unsigned shmax = ql == 0 ? 24 : 32 - 8 * ql;
#pragma unroll 1
      while (m) {
        const unsigned q8 = __builtin_ctzll(m), s = (taken >> (q8 >> 1)) & 7;
        taken += 1u << (q8 >> 1);
        if (s == 7) open &= ~(1ull << q8);
        asm volatile("v_addc_co_u32 %0, vcc, 0, %0, %1" : "+v"(thr) : "s"(0xFFull << q8) : "vcc");
        vaux = vpoll + (thr & 7) * 32;
        dnext |= (0x80 + q8 + s) << sh;
        sh += 8;
        if (sh >= shmax) break;
        m = ready(c);
      }
      lds_st<unsigned>(vcw, dnext);
      vcw ^= 4;
      if (open) poll();
    } else {
      d = rfl(c);
      unsigned e = d;
#pragma unroll 1
      while (e & 0x80) {
        asm volatile("s_nop 4\n\ts_mov_b32 m0, %0\n\ts_nop 0\n\t"
                     "buffer_load_dwordx4 %2, %4, %5 offen " LDAUX " lds\n\t"
                     "buffer_load_dwordx4 %2, %4, %5 offen offset:1024 " LDAUX " lds\n\t"
                     "buffer_load_dwordx4 %2, %4, %5 offen offset:2048 " LDAUX " lds\n\t"
                     "buffer_load_dwordx4 %2, %4, %5 offen offset:3072 " LDAUX " lds\n\t"
                     "s_mov_b32 m0, %1\n\ts_nop 0\n\t"
                     "buffer_load_dwordx4 %3, %4, %5 offen " LDAUX " lds"
                     :: "s"(mdma + stt), "s"(maux + stt), "v"(vdma), "v"(vaux), "s"(rdat), "s"((e & 63) * STEP_BYTES) : "memory");
        stt = stt == 3 * STAGE_BYTES ? 0 : stt + STAGE_BYTES;
        ++ql;
        e >>= 8;
      }
      vcell ^= 4;
    }
    if (!decltype(busy)::value && d == CELL_ABORT) { aborted = 1; left = 1; ql = 1; }
    if (decltype(busy)::value) {
      st = st == 3 * STAGE_BYTES ? 0 : st + STAGE_BYTES;
      ra = va + st; rb = vb + st;
    }
    asm volatile("" : "+v"(ra), "+v"(rb), "+v"(vcell));
  };
  auto landed = [&]() {
    asm volatile("s_waitcnt vmcnt(10)\n\ts_cmp_ge_u32 %0, 3\n\ts_cbranch_scc1 .Lwv%=\n\t"
                 "s_waitcnt vmcnt(5)\n\ts_cmp_ge_u32 %0, 2\n\ts_cbranch_scc1 .Lwv%=\n\t"
                 "s_waitcnt vmcnt(0)\n"
                 ".Lwv%=:" :: "s"(ql) : "scc", "memory");
  };
  using T = std::integral_constant<bool, true>; using F = std::integral_constant<bool, false>;
  if (grp == 1) BARRIER();
  auto mainloop = [&](auto is_poller) {
  i32x4 a[9], b[4];
#pragma unroll
  for (int i = 0; i < PF; ++i) a[i] = i32x4{0, 0, 0, 0};
#pragma unroll 1
  while (true) {
#pragma unroll 1
    while (ql == 0) {
      unsigned c = first_read();
      asm volatile("s_waitcnt lgkmcnt(0)" : "+v"(c) :: "memory");
      issue(c, F{}, is_poller);
      if (!decltype(is_poller)::value && grp == 1) landed();
      WAIT_LGKM(0);
      BARRIER();
      if (grp == 0) landed();
      if (left == ql) WAIT_VM(0);
      BARRIER();
#pragma unroll
      for (int i = 0; i < PF; ++i) a[i] = lds_ld<i32x4>(ra + i * 1024);
      SCHED();
    }
    if (left == ql) break;
    --ql;
    PH(0);
    unsigned c = first_read();
#pragma unroll
    for (int j = 0; j < 4; ++j) b[j] = lds_ld<i32x4>(rb + j * 1024);
#pragma unroll
    for (int i = PF; i < 9; ++i) a[i] = lds_ld<i32x4>(ra + i * 1024);
    SCHED();
    asm volatile("s_waitcnt lgkmcnt(%1)" : "+v"(c) : "n"(13 - PF) : "memory");
    issue(c, T{}, is_poller);
    PH(1);
    if (!decltype(is_poller)::value) landed();
    WAIT_LGKM(0);
    PH(2);
    SCHED();
    BARRIER();
    PH(3);
    SCHED();
    __builtin_amdgcn_s_setprio(1);
#pragma unroll
    for (int i = 0; i < 8; ++i) {
#pragma unroll
      for (int j = 0; j < 4; ++j) acc[i][j] = __builtin_amdgcn_mfma_i32_16x16x64_i8(a[i], b[j], acc[i][j], 0, 0, 0);
      if (i >= PD && i - PD < PF) { SCHED(); a[i - PD] = lds_ld<i32x4>(ra + (i - PD) * 1024); SCHED(); }
    }
    if (nine) {
#pragma unroll
      for (int j = 0; j < 4; ++j) acc[8][j] = __builtin_amdgcn_mfma_i32_16x16x64_i8(a[8], b[j], acc[8][j], 0, 0, 0);
    }
    __builtin_amdgcn_s_setprio(0);
    SCHED();
    PH(4);
    --left;
    if (left == ql && grp == 1) WAIT_VM(0);
    BARRIER();
  }
  };
  if (poller) mainloop(T{}); else mainloop(F{});
  if (grp == 0) { WAIT_VM(0); BARRIER(); __builtin_amdgcn_s_setprio(2); }
  STAMP(3);
  STAMPV(5, n);
  STAMPV(7, __builtin_amdgcn_s_memtime());
  unsigned tz = threadIdx.x; asm volatile("" : "+v"(tz));
  const unsigned ln = tz & 63;
  float lo = 0.f;
  if (aborted) lo = __builtin_inff();
  const rsrc_t ro = mk_rsrc(outp, MN * MN * 4);
  const unsigned vo = (ln >> 4) * 4 * (MN * 4) + (ln & 15) * 16;
  const unsigned so0 = (f0 + grp * 9) * 16 * (MN * 4) + (tj * 256 + wn * 64) * 4;
  const unsigned so7 = grp == 0 ? so0 : (240 + ti - 7) * 16 * (MN * 4) + (tj * 256 + wn * 64) * 4;
  auto store_row = [&](const int i) {
#pragma unroll
    for (int r = 0; r < 4; ++r) {
      f32x4 y;
#pragma unroll
      for (int j = 0; j < 4; ++j) y[j] = fmaxf((float)acc[i][j][r] * scale, lo);
      st128(y, ro, vo, (i < 7 ? so0 : so7) + (i * 16 + r) * (MN * 4));
    }
  };
  i32x4 a[3], b[4];
  __builtin_amdgcn_s_waitcnt(0xC07F);
  a[0] = lds_ld<i32x4>(ra);
  SCHED();
#pragma unroll
  for (int j = 0; j < 4; ++j) { b[j] = lds_ld<i32x4>(rb + j * 1024); SCHED(); }
  a[1] = lds_ld<i32x4>(ra + 1024);
#pragma unroll
  for (int c = 0; c < 4; ++c) {
    unsigned s2 = st, k = left;
#pragma unroll 1
    do {
      const unsigned nx = s2 == 3 * STAGE_BYTES ? 0 : s2 + STAGE_BYTES;
      unsigned add;
      asm("s_cmp_eq_u32 %2, 1\n\ts_cselect_b32 %0, %3, %4\n\ts_cselect_b32 %1, 0x800, 0" : "=&s"(s2), "=&s"(add) : "s"(k), "s"(st), "s"(nx) : "scc");
      const unsigned fa = va + s2, fc = fa + add, fb = vb + s2;
      SCHED();
#pragma unroll
      for (int j = 0; j < 4; ++j) acc[2 * c][j] = __builtin_amdgcn_mfma_i32_16x16x64_i8(a[0], b[j], acc[2 * c][j], 0, 0, 0);
      SCHED();
      a[0] = lds_ld<i32x4>(fc + 2 * c * 1024);
      SCHED();
      if (c == 3 && nine) {
#pragma unroll
        for (int j = 0; j < 4; ++j) acc[8][j] = __builtin_amdgcn_mfma_i32_16x16x64_i8(a[2], b[j], acc[8][j], 0, 0, 0);
      }
#pragma unroll
      for (int j = 0; j < 4; ++j) {
        acc[2 * c + 1][j] = __builtin_amdgcn_mfma_i32_16x16x64_i8(a[1], b[j], acc[2 * c + 1][j], 0, 0, 0);
        SCHED();
        b[j] = lds_ld<i32x4>(fb + j * 1024);
        SCHED();
      }
      if (c >= 2) a[2] = lds_ld<i32x4>(fa + 8 * 1024);
      a[1] = lds_ld<i32x4>(fc + (2 * c + 1) * 1024);
      SCHED();
    } while (--k);
    store_row(2 * c);
    store_row(2 * c + 1);
    if (c == 3 && nine) store_row(8);
  }
}

static const char* const LATE_ORDER[NRANK] = {
  "fAhGOfjcJJOeLmNbmkIKBACNpfdGmKoHDAhbPkapdMHnDeaFJNlinLLgOdFOggGpHjnFcgLjaIPIdhMCBcFMbEAlDJepcioEjKhGCmlPkNEoiMnBHaflebKIoBiPEkDC",
  "GicGkdOphdgCnijpDjLhgmpPFjIOAooAeKJMcFLNPFejMecmBOCINKNHLAnGaMbJaDdhKlPHNknodBcamfLbgnpfICfEMlHgHmGOFhJEliKkAfPeDBbEboalkDJIBiEC",
  "KhfaFBFnCOLpAehipmHAgGNdMogDgkjjmEaiOhOlfJneEPapcpcLGAekFKdHfNlbCKMkjILIDoHOJMdPBcjNnPGcmnGbdJeFLDfMAgNiIHbmalIBPCkhoJKEboiDClBE",
  "hFpeGmnpEcCLGjaMddNgKOlolOjlLMngfkcJfDBDeHCpNMBHKFJbaHInAgmEecLbGmjFkFAfOIPciaPJNdhAgEBCPKiPhIokDjobHipLMnhdONGebmaAfJkIKlBiCoED",
  "HgCpLdcbjeJhiMNfpkaPaAAjBmnNjgckGMOBhdbFFJLFnKjNahKeHhIfCOpPHpMIODlLoiDGlcnKEIfGkeEMOcgomiodCdAmlPBNbaEgLJFAHGmKJeInPlDfboCEkBDi",
  "JeOAgLKehPoaAFFFKpEbpLEdGOcNJdekpgfmJOfhjcmnHkHGcaKBIColgNmiAjMMdIHdPeNOpManDlbLgNLjihGPmjAGMBbDaInfkHhfclKnCiFPJoklDCiBIoECbBDE",
  "cHPjAaLmkpjMjoJFGCDEdIBOMfbLfKmiAagDJdNkpGAnHGdcOhoceHjaIghibpFpNEBnONlIlMHdmBPfLcgJhgPFnhFLiKeaCkMlKCeAGbOmDJEoIKfNEenkClPDBobi",
  "dIFMKhmjMLIKjGkdphgEnhJiAgHlfOPHNmlfapOLElOmdfoHhLenLeoNFiJBgkONnPJccDpgjjMGCaAGHeIFpeAGKbcNMFAacbdmanPIBJfDCDkbiKoECPiBbEkCloBD",
};
static void build_schedule(Params& p) {
  static const int first[8] = {0, 1, 0, 1, 2, 3, 2, 3};
  int cnt[32] = {0};
  for (int k = 0; k < 256; ++k) {
    const int r = k >> 3;
    int pr = (k & 7) + 8 * (r < 8 ? first[r] : r & 3);
    if (k >= 128) { const char c = LATE_ORDER[p.rank][k - 128]; pr = c >= 'a' ? c - 'a' : c - 'A' + 16; }
    if (pr >= 16) pr = 16 + 4 * (pr & 3) + ((pr >> 2) & 3);
    p.sched[2 * k] = (unsigned short)(2 * pr | cnt[pr] << 6);
    p.sched[2 * k + 1] = (unsigned short)((2 * pr + 1) | cnt[pr] << 6);
    ++cnt[pr];
  }
}

extern "C" void kernel_launch(int rank, int nranks, ncclComm_t comm,
                              hipStream_t stream,
                              void* const* d_in, const int* in_sizes, int n_in,
                              void* d_out, int out_size,
                              const int* in_dtypes, int out_dtype,
                              void* const* d_out_all,
                              void* const* d_ws_all, size_t ws_bytes) {
  if (nranks != NRANK || (unsigned)rank >= (unsigned)NRANK || n_in != 4 || in_sizes[0] != (int)SHARD_BYTES || in_sizes[1] != (int)SHARD_BYTES ||
      out_size != (int)(MN * MN) || ws_bytes < (size_t)CTRL_OFF + (1u << 20)) {
    fprintf(stderr, "unexpected problem shape\n");
    return;
  }
  HIP_CHECK(hipFuncSetAttribute((const void*)fused_gemm_allgather, hipFuncAttributeMaxDynamicSharedMemorySize, LDS_BYTES));
  Params p;
  p.x = (const char*)d_in[0]; p.w = (const char*)d_in[1]; p.sx = (const float*)d_in[2]; p.sw = (const float*)d_in[3];
  p.out = (float*)d_out;
  p.me = (char*)d_ws_all[rank];
  for (int i = 0; i < 14; ++i) p.peer[i] = (char*)d_ws_all[(rank + 1 + i % 7) % NRANK];
  p.rank = rank;
  build_schedule(p);
  hipLaunchKernelGGL(fused_gemm_allgather, dim3(NWG), dim3(512), LDS_BYTES, stream, p);
  HIP_CHECK(hipGetLastError());
}
```
